```python
import numpy as np

kernel_name = "gpu_gemm_a2a_m4096_k4096_n16384_e4m3e5m2_dqrelu"


ABSMAX_RTOL = 0.05

COMPUTE_DTYPE = "float32"
EPILOGUE = "dequant_relu"
IN_QUANT = None
OUT_QUANT = None
IN_FP8 = "e4m3e5m2"
IN_DTYPES = {"x": "e4m3", "w_mat": "e5m2",
             "scale_x": "float32", "scale_w": "float32"}
WORLD_SIZE = 8

def _snap_fp8(x, mant_bits, max_abs, min_normal):
    a = np.abs(x.astype(np.float64))
    _m, _e = np.frexp(a)
    q = np.where(a >= min_normal, np.ldexp(1.0, _e - 1 - mant_bits),
                 min_normal * 2.0 ** -mant_bits)
    snapped = np.minimum(np.round(a / q) * q, max_abs)
    return (np.sign(x) * snapped).astype(np.float32)

def setup_inputs():

    x = _snap_fp8(np.random.randn(4096, 4096).astype(np.float32), 3, 448.0, 2.0 ** -6)
    w_mat = _snap_fp8(np.random.randn(4096, 16384).astype(np.float32), 2, 57344.0, 2.0 ** -14)
    scale_x = (0.02 * (1.0 + np.random.rand(1))).astype(np.float32)
    scale_w = (0.02 * (1.0 + np.random.rand(1))).astype(np.float32)
    return {"x": x, "w_mat": w_mat, "scale_x": scale_x, "scale_w": scale_w}

def reference(x, w_mat, scale_x, scale_w):
    acc = (x @ w_mat).astype(np.float32)
    y = (acc * (scale_x * scale_w)).astype(np.float32)
    return np.maximum(y, 0.0).astype(np.float32)
```

```cpp
#include <hip/hip_runtime.h>
#include <rccl/rccl.h>
#include <cstdio>
#include <cstdint>

#define HIP_CHECK(expr) do { hipError_t _e = (expr); if (_e != hipSuccess) { \
    fprintf(stderr, "HIP error %s at %s:%d\n", hipGetErrorString(_e), __FILE__, __LINE__); \
    return; } } while (0)

#define OUT_ST "nt"
namespace {
typedef int v8i __attribute__((ext_vector_type(8)));
typedef int v4i __attribute__((ext_vector_type(4)));
typedef int v2i __attribute__((ext_vector_type(2)));
typedef float v4f __attribute__((ext_vector_type(4)));
typedef unsigned v4u __attribute__((ext_vector_type(4)));
#define LAS __attribute__((address_space(3)))

constexpr int NR = 8;
constexpr int M = 4096, K = 4096, NT = 16384, NS = NT / NR, MS = M / NR;
constexpr int BM = 256, BN = 128, BK = 128, NSTEP = K / BK;
constexpr int XB = BM * BK, WB = BK * BN, STAGE = XB + WB, NSTAGE = 3;
constexpr int SYNC_OFF = STAGE * NSTAGE;
constexpr int DUMMY_OFF = SYNC_OFF + 128;
constexpr int LDS_BYTES = DUMMY_OFF + 1024;
constexpr int NTHREADS = 768, NWG = 256;
constexpr int STEP_B = MS * BK;
constexpr size_t SHARD_B = (size_t)MS * K;
constexpr size_t FLAG_OFF = 2 * SHARD_B;
constexpr size_t CNT_OFF = FLAG_OFF + NR * 1024;
constexpr size_t INV_OFF = CNT_OFF + (size_t)NWG * 128;
constexpr size_t TMO_OFF = INV_OFF + 16 * 128;
constexpr size_t WS_NEED = TMO_OFF + 4096;
#define LEAD 0
constexpr unsigned long long DEADLINE_TICKS = 50000000ull;
constexpr unsigned LDS_SPIN_CAP = 1u << 26;

struct Params {
  const unsigned char* x;
  const unsigned char* w;
  const float* sx;
  const float* sw;
  float* out;
  unsigned char* myws;
  unsigned char* ws[NR];
  int rank;
};

#define WAITV(n) asm volatile("s_waitcnt vmcnt(" #n ")" ::: "memory")
#define WAITL(n) asm volatile("s_waitcnt lgkmcnt(" #n ")" ::: "memory")

__device__ __forceinline__ void dma16(unsigned m0v, int voff, __amdgpu_buffer_rsrc_t srd, int soff) {
  asm volatile("s_mov_b32 m0, %0\n\ts_nop 0\n\tbuffer_load_dwordx4 %1, %2, %3 offen lds" :: "s"(m0v), "v"(voff), "s"(srd), "s"(soff) : "memory");
}
__device__ __forceinline__ void dma16x4(unsigned m0v, int v0, int v1, int v2, int v3, __amdgpu_buffer_rsrc_t srd, int soff) {
  asm volatile("s_mov_b32 m0, %0\n\ts_nop 0\n\t"
               "buffer_load_dwordx4 %1, %5, %6 offen lds\n\t"
               "buffer_load_dwordx4 %2, %5, %6 offen offset:1024 lds\n\t"
               "buffer_load_dwordx4 %3, %5, %6 offen offset:2048 lds\n\t"
               "buffer_load_dwordx4 %4, %5, %6 offen offset:3072 lds"
               :: "s"(m0v), "v"(v0), "v"(v1), "v"(v2), "v"(v3), "s"(srd), "s"(soff) : "memory");
}
__device__ __forceinline__ void dma16_sc(unsigned m0v, int voff, __amdgpu_buffer_rsrc_t srd, int soff) {
  asm volatile("s_mov_b32 m0, %0\n\ts_nop 0\n\tbuffer_load_dwordx4 %1, %2, %3 offen sc0 sc1 lds" :: "s"(m0v), "v"(voff), "s"(srd), "s"(soff) : "memory");
}
__device__ __forceinline__ void st16_sys(unsigned char* p, v4u v) {
#define PUSH_ST "sc0 sc1"
  asm volatile("global_store_dwordx4 %0, %1, off " PUSH_ST "\n\ts_nop 1" :: "v"(p), "v"(v) : "memory");
}
__device__ __forceinline__ void st4_sys(unsigned char* p, unsigned v) {
  asm volatile("global_store_dword %0, %1, off sc0 sc1" :: "v"(p), "v"(v) : "memory");
}
__device__ __forceinline__ v4u ld16_asm(const unsigned char* p) {
  v4u v;
  asm volatile("global_load_dwordx4 %0, %1, off" : "=v"(v) : "v"(p) : "memory");
  return v;
}
__device__ __forceinline__ int rfl(int v) { return __builtin_amdgcn_readfirstlane(v); }
__device__ __forceinline__ int lds_ld(LAS int* p) { return *(volatile LAS int*)p; }
__device__ __forceinline__ void lds_st(LAS int* p, int v) { *(volatile LAS int*)p = v; }

__global__ void __launch_bounds__(NTHREADS) fused_ag_gemm(Params p) {
  extern __shared__ __attribute__((aligned(16))) unsigned char smem[];
  LAS unsigned char* lds = (LAS unsigned char*)smem;
  LAS int* sync = (LAS int*)(lds + SYNC_OFF);
  const int tid = threadIdx.x, lane = tid & 63;
  const int wid = rfl(tid >> 6);
  const int bid = blockIdx.x;
  unsigned char* const myws = p.myws;
  const int srcr = bid & 7, idx = bid >> 3;
  const int rb = idx & 1, tn = idx >> 1;
  const int m0 = (srcr * 2 + rb) * BM, n0 = tn * BN;
  const bool remote = srcr != p.rank;
  unsigned* const cnt = (unsigned*)(myws + CNT_OFF + (size_t)bid * 128);
  const unsigned prev = __hip_atomic_load(cnt, __ATOMIC_RELAXED, __HIP_MEMORY_SCOPE_AGENT);
  unsigned char* fp = p.ws[lane & (NR - 1)];
  v4u piece = {0u, 0u, 0u, 0u};
  if (wid < 8) piece = *(const v4u*)(p.x + (size_t)((bid & 7) * 64 + wid * 8 + (lane >> 3)) * K + (bid >> 3) * BK + (lane & 7) * 16);
  const unsigned ldsa = (unsigned)(size_t)lds;
  if (tid < 16) sync[tid] = 0;
  const unsigned call = (unsigned)rfl((int)prev) + 1u;
  asm volatile("" : "+v"(fp));
  if (wid < 8) {
    st16_sys(myws + (size_t)(call & 1u) * SHARD_B + (size_t)(bid >> 3) * STEP_B + ((bid & 7) * 64 + wid * 8) * BK + lane * 16, piece);
    WAITV(0);
  } else if (wid == 11 && idx == 0) {
    __builtin_amdgcn_fence(__ATOMIC_ACQUIRE, "");
  }
  WAITL(0);
  __builtin_amdgcn_s_barrier();
  if (wid == 7) {
    if (lane < NR && lane != p.rank) st4_sys(fp + FLAG_OFF + p.rank * 1024 + bid * 4, call);
    if (lane == 0) __hip_atomic_store(cnt, call, __ATOMIC_RELAXED, __HIP_MEMORY_SCOPE_AGENT);
  }
  const unsigned char* const xsrc = p.ws[srcr] + (size_t)(call & 1u) * SHARD_B;
  if (wid == 11) {
    const unsigned xcc = __builtin_amdgcn_s_getreg(20 | (3 << 11)) & 15u;
    unsigned* const invw = (unsigned*)(myws + INV_OFF + xcc * 128);
    if (idx == 0) {
      WAITV(0);
      if (lane == 0) __hip_atomic_store(invw, call, __ATOMIC_RELAXED, __HIP_MEMORY_SCOPE_AGENT);
    }
    if (!remote) return;
    const unsigned long long* fl = (const unsigned long long*)(myws + FLAG_OFF + srcr * 1024) + lane * 2;
    unsigned* const tmo = (unsigned*)(myws + TMO_OFF);
    auto rs = __builtin_amdgcn_make_buffer_rsrc((void*)xsrc, 0, (int)SHARD_B, 0x00020000);
    const unsigned long long t0 = __builtin_amdgcn_s_memrealtime();
    bool inv_ok = idx == 0;
    int pf = rb ? 1 : LEAD + 1;
    for (unsigned it = 0;; ++it) {
      const unsigned long long f0 = __hip_atomic_load(fl, __ATOMIC_RELAXED, __HIP_MEMORY_SCOPE_SYSTEM);
      const unsigned long long f1 = __hip_atomic_load(fl + 1, __ATOMIC_RELAXED, __HIP_MEMORY_SCOPE_SYSTEM);
      const unsigned iv = inv_ok ? call : __hip_atomic_load(invw, __ATOMIC_RELAXED, __HIP_MEMORY_SCOPE_AGENT);
      const bool ok = (int)((unsigned)f0 - call) >= 0 && (int)((unsigned)(f0 >> 32) - call) >= 0 &&
                      (int)((unsigned)f1 - call) >= 0 && (int)((unsigned)(f1 >> 32) - call) >= 0;
      unsigned long long okm = __ballot(ok);
      inv_ok = (int)((unsigned)rfl((int)iv) - call) >= 0;
      if (!inv_ok && okm != 0ull && __builtin_amdgcn_s_memrealtime() - t0 > 500ull) {
        __builtin_amdgcn_fence(__ATOMIC_ACQUIRE, "");
        WAITV(0);
        inv_ok = true;
      }
      if ((it & 15u) == 15u &&
          (rfl((int)__hip_atomic_load(tmo, __ATOMIC_RELAXED, __HIP_MEMORY_SCOPE_AGENT)) == (int)call ||
           __builtin_amdgcn_s_memrealtime() - t0 > DEADLINE_TICKS)) {
        if (lane == 0) { __hip_atomic_store(tmo, call, __ATOMIC_RELAXED, __HIP_MEMORY_SCOPE_AGENT); lds_st(sync + 3, 1); }
        okm = ~0ull; inv_ok = true; pf = LEAD + 1;
      }
      if (inv_ok) {
        while (pf <= LEAD && ((okm >> (2 * pf)) & 1ull)) {
          const int po = rfl(pf) * STEP_B + tn * 2048;
          dma16(ldsa + DUMMY_OFF, lane * 16, rs, po);
          dma16(ldsa + DUMMY_OFF, lane * 16, rs, po + 1024);
          ++pf;
        }
        if (pf > LEAD) {
          if (lane == 0) { lds_st(sync + 12, (int)(unsigned)okm); lds_st(sync + 13, (int)(unsigned)(okm >> 32)); }
          if (okm == ~0ull) break;
        }
      }
    }
    WAITV(0);
    return;
  }
  if (wid >= 8) {
    const int b = wid - 8;
    const int rstride = remote ? BK : K, sstride = remote ? STEP_B : BK;
    const unsigned char* xb = remote ? xsrc + (size_t)rb * BM * BK : p.x + (size_t)rb * BM * K;
    auto rx = __builtin_amdgcn_make_buffer_rsrc((void*)xb, 0, (int)SHARD_B - rb * BM * rstride, 0x00020000);
    auto rw = __builtin_amdgcn_make_buffer_rsrc((void*)(p.w + p.rank * NS + n0), 0, K * NT - (p.rank * NS + n0), 0x00020000);
    const int lr = lane >> 3, lc = lane & 7;
    const int sw0 = (lc ^ (lr >> 1)) << 4, sw1 = (lc ^ (4 + (lr >> 1))) << 4;
    const int xv0 = lr * rstride + sw0, xv1 = (lr + 8) * rstride + sw1 - 1024;
    const int xv2 = (lr + 16) * rstride + sw0 - 2048, xv3 = (lr + 24) * rstride + sw1 - 3072;
    const int wv0 = lr * NT + sw0, wv1 = (lr + 8) * NT + sw0 - 1024;
    const int wv2 = (lr + 16) * NT + sw1 - 2048, wv3 = (lr + 24) * NT + sw1 - 3072;
#define LPRIO 2
    __builtin_amdgcn_s_setprio(LPRIO);
    const unsigned ldsb = ldsa + b * STAGE;
    const int send = (remote && rb == 0 && LEAD > 1) ? NSTEP + LEAD - 1 : NSTEP;
    for (int s = b; s < send; s += NSTAGE) {
      if (s >= NSTAGE) {
        for (unsigned it = 0; it < LDS_SPIN_CAP; ++it) {
          const int d = lds_ld(sync + 4 + (lane & 7));
          if (__all(d >= s - 2)) break;
          __builtin_amdgcn_s_sleep(1);
        }
      }
      if (s < NSTEP) {
#pragma unroll
        for (int qq = 0; qq < 4; ++qq) dma16x4(ldsb + XB + qq * 4096, wv0, wv1, wv2, wv3, rw, (s * BK + qq * 32) * NT);
      }
      if (remote) {
        const int ub = s < NSTEP ? 2 * s + rb : 2 * (s - LEAD + 1) + 1;
        for (unsigned it = 0; it < LDS_SPIN_CAP; ++it) {
          const int v = lds_ld(sync + 12 + (lane & 3));
          const unsigned mw = (unsigned)__builtin_amdgcn_readlane(v, ub >> 5);
          if (((mw >> (ub & 31)) & 1u) && __builtin_amdgcn_readlane(v, 2) >= s) break;
          __builtin_amdgcn_s_sleep(1);
        }
      }
      if (s >= NSTEP) {
        const int po = (s - LEAD + 1) * STEP_B + BM * BK + tn * 2048;
        dma16(ldsa + DUMMY_OFF, lane * 16, rx, po);
        dma16(ldsa + DUMMY_OFF, lane * 16, rx, po + 1024);
        if (lane == 0) lds_st(sync + 14, s + 1);
        break;
      }
      const int so = s * sstride;
#pragma unroll 2
      for (int pp = 0; pp < 8; ++pp) {
        const int pr = (pp + tn) & 7;
        dma16x4(ldsb + pr * 4096, xv0, xv1, xv2, xv3, rx, so + pr * 32 * rstride);
      }
      if (lane == 0) lds_st(sync + 14, s + 1);
      WAITV(0);
      if (lane == 0) lds_st(sync + b, s + 1);
    }
    WAITV(0);
    return;
  }
  const int wm = wid & 3, wn = wid >> 2;
  const int r = lane & 15, g = lane >> 4;
  const int xa0 = (wm * 64 + r) * 128 + ((g ^ ((r >> 1) & 7)) << 4);
  const int xa1 = xa0 ^ 64;
  const int fw = ((r >> 2) & 3) | ((g & 1) << 2);
  const int wa = XB + (16 * g + (r >> 1)) * 128 + (r & 1) * 8;
  int wan[4];
#pragma unroll
  for (int nt = 0; nt < 4; ++nt) wan[nt] = wa + (((wn * 4 + nt) ^ fw) << 4);
  v4f acc[4][4];
#pragma unroll
  for (int i = 0; i < 4; ++i)
#pragma unroll
    for (int j = 0; j < 4; ++j) acc[i][j] = (v4f){0.f, 0.f, 0.f, 0.f};
  int buf = 0;
  int fullv = 0;
  for (int s = 0; s < NSTEP; ++s) {
    if (rfl(fullv) <= s) {
      for (unsigned it = 0; it < LDS_SPIN_CAP; ++it) {
        fullv = lds_ld(sync + buf);
        if (rfl(fullv) > s) break;
        __builtin_amdgcn_s_sleep(1);
      }
    }
    const int sb = buf * STAGE;
    v8i xf[4], wf[4];
#pragma unroll
    for (int nt = 0; nt < 4; ++nt) {
      v2i q[4];
#pragma unroll
      for (int hs = 0; hs < 4; ++hs)
        q[hs] = __builtin_amdgcn_ds_read_tr8_b64_v2i32((LAS v2i*)(lds + sb + wan[nt] + (hs >> 1) * 8192 + (hs & 1) * 1024));
      wf[nt] = (v8i){q[0].x, q[0].y, q[1].x, q[1].y, q[2].x, q[2].y, q[3].x, q[3].y};
    }
#pragma unroll
    for (int mt = 0; mt < 4; ++mt) {
      v4i a = *(LAS v4i*)(lds + sb + xa0 + mt * 2048), c = *(LAS v4i*)(lds + sb + xa1 + mt * 2048);
      xf[mt] = (v8i){a.x, a.y, a.z, a.w, c.x, c.y, c.z, c.w};
    }
    buf = buf + 1; if (buf >= NSTAGE) buf = 0;
    __builtin_amdgcn_s_waitcnt(0xC07F);
    __builtin_amdgcn_sched_barrier(0);
    lds_st(sync + 4 + wid, s + 1);
    fullv = lds_ld(sync + buf);
    __builtin_amdgcn_sched_barrier(0);
    __builtin_amdgcn_s_setprio(1);
#pragma unroll
    for (int mt = 0; mt < 4; ++mt)
#pragma unroll
      for (int nt = 0; nt < 4; ++nt)
        acc[mt][nt] = __builtin_amdgcn_mfma_scale_f32_16x16x128_f8f6f4(wf[nt], xf[mt], acc[mt][nt], 1, 0, 0, 0, 0, 0);
    __builtin_amdgcn_s_setprio(0);
    __builtin_amdgcn_sched_barrier(0);
  }
  for (unsigned it = 0; it < LDS_SPIN_CAP; ++it) {
    const int d = lds_ld(sync + 4 + (lane & 7));
    if (__all(d >= NSTEP - 2)) break;
    __builtin_amdgcn_s_sleep(1);
  }
  float sc = p.sx[0] * p.sw[0];
  const bool bad = rfl(lds_ld(sync + 3)) != 0;
  LAS unsigned char* eb = lds + ((NSTEP - 3) % NSTAGE) * STAGE + wid * 4608;
  float* o = p.out + (size_t)(m0 + wm * 64 + g) * NS + n0 + wn * 64 + r * 4;
#pragma unroll
  for (int mt = 0; mt < 4; ++mt) {
#pragma unroll
    for (int nt = 0; nt < 4; ++nt) {
      v4f v = acc[mt][nt] * sc;
      v.x = fmaxf(v.x, 0.f); v.y = fmaxf(v.y, 0.f); v.z = fmaxf(v.z, 0.f); v.w = fmaxf(v.w, 0.f);
      if (bad) { const float q = __builtin_nanf(""); v = (v4f){q, q, q, q}; }
      *(LAS v4f*)(eb + r * 272 + nt * 64 + g * 16) = v;
    }
#pragma unroll
    for (int i = 0; i < 4; ++i) {
      const v4f v = *(LAS v4f*)(eb + (i * 4 + g) * 272 + r * 16);
      asm volatile("global_store_dwordx4 %0, %1, off " OUT_ST "\n\ts_nop 1" :: "v"(o + (size_t)(mt * 16 + i * 4) * NS), "v"(v) : "memory");
    }
  }
}
}

extern "C" void kernel_launch(int rank, int nranks, ncclComm_t comm,
                              hipStream_t stream,
                              void* const* d_in, const int* in_sizes, int n_in,
                              void* d_out, int out_size,
                              const int* in_dtypes, int out_dtype,
                              void* const* d_out_all,
                              void* const* d_ws_all, size_t ws_bytes) {
  (void)comm; (void)in_sizes; (void)n_in; (void)out_size; (void)in_dtypes; (void)out_dtype; (void)d_out_all;
  if (nranks != NR || (unsigned)rank >= (unsigned)NR || ws_bytes < WS_NEED) {
    fprintf(stderr, "unexpected configuration\n");
    return;
  }
  Params p;
  p.x = (const unsigned char*)d_in[0];
  p.w = (const unsigned char*)d_in[1];
  p.sx = (const float*)d_in[2];
  p.sw = (const float*)d_in[3];
  p.out = (float*)d_out;
  for (int i = 0; i < NR; ++i) p.ws[i] = (unsigned char*)d_ws_all[i];
  p.myws = p.ws[rank];
  p.rank = rank;
  HIP_CHECK(hipFuncSetAttribute((const void*)fused_ag_gemm, hipFuncAttributeMaxDynamicSharedMemorySize, LDS_BYTES));
  fused_ag_gemm<<<dim3(NWG), dim3(NTHREADS), LDS_BYTES, stream>>>(p);
  HIP_CHECK(hipGetLastError());
}
```

```cpp
#include <hip/hip_runtime.h>
#include <rccl/rccl.h>
#include <cstdio>
#include <cstdint>

#define HIP_CHECK(expr) do { hipError_t _e = (expr); if (_e != hipSuccess) { \
    fprintf(stderr, "HIP error %s at %s:%d\n", hipGetErrorString(_e), __FILE__, __LINE__); \
    return; } } while (0)

#define OUT_ST "nt"
namespace {
typedef int v8i __attribute__((ext_vector_type(8)));
typedef int v4i __attribute__((ext_vector_type(4)));
typedef int v2i __attribute__((ext_vector_type(2)));
typedef float v4f __attribute__((ext_vector_type(4)));
typedef unsigned v4u __attribute__((ext_vector_type(4)));
#define LAS __attribute__((address_space(3)))

constexpr int NR = 8;
constexpr int M = 4096, K = 4096, NT = 16384, NS = NT / NR, MS = M / NR;
constexpr int BM = 256, BN = 128, BK = 128, NSTEP = K / BK;
constexpr int XB = BM * BK, WB = BK * BN, STAGE = XB + WB, NSTAGE = 3;
constexpr int SYNC_OFF = STAGE * NSTAGE;
constexpr int DUMMY_OFF = SYNC_OFF + 128;
constexpr int LDS_BYTES = DUMMY_OFF + 1024;
constexpr int NTHREADS = 768, NWG = 256;
constexpr int STEP_B = MS * BK;
constexpr size_t SHARD_B = (size_t)MS * K;
constexpr size_t INBOX_OFF = 2 * SHARD_B;
constexpr size_t FLAG_OFF = INBOX_OFF + 2 * NR * (size_t)STEP_B;
constexpr int FLAG_B = 2048;
constexpr size_t CNT_OFF = FLAG_OFF + NR * FLAG_B;
#define PUSH0 0
constexpr size_t INV_OFF = CNT_OFF + (size_t)NWG * 128;
constexpr size_t TMO_OFF = INV_OFF + 16 * 128;
constexpr size_t WS_NEED = TMO_OFF + 4096;
#define NSPLIT 3
constexpr unsigned long long DEADLINE_TICKS = 50000000ull;
constexpr unsigned LDS_SPIN_CAP = 1u << 26;

struct Params {
  const unsigned char* x;
  const unsigned char* w;
  const float* sx;
  const float* sw;
  float* out;
  unsigned char* myws;
  unsigned char* ws[NR];
  unsigned char* peer[NR - 1];
  int rank;
};

#define WAITV_(n) asm volatile("s_waitcnt vmcnt(" #n ")" ::: "memory")
#define WAITV(n) WAITV_(n)
#define WAITL(n) asm volatile("s_waitcnt lgkmcnt(" #n ")" ::: "memory")

__device__ __forceinline__ void dma16(unsigned m0v, int voff, __amdgpu_buffer_rsrc_t srd, int soff) {
  asm volatile("s_mov_b32 m0, %0\n\ts_nop 0\n\tbuffer_load_dwordx4 %1, %2, %3 offen lds" :: "s"(m0v), "v"(voff), "s"(srd), "s"(soff) : "memory");
}
__device__ __forceinline__ void dma16x4(unsigned m0v, int v0, int v1, int v2, int v3, __amdgpu_buffer_rsrc_t srd, int soff) {
  asm volatile("s_mov_b32 m0, %0\n\ts_nop 0\n\t"
               "buffer_load_dwordx4 %1, %5, %6 offen lds\n\t"
               "buffer_load_dwordx4 %2, %5, %6 offen offset:1024 lds\n\t"
               "buffer_load_dwordx4 %3, %5, %6 offen offset:2048 lds\n\t"
               "buffer_load_dwordx4 %4, %5, %6 offen offset:3072 lds"
               :: "s"(m0v), "v"(v0), "v"(v1), "v"(v2), "v"(v3), "s"(srd), "s"(soff) : "memory");
}
__device__ __forceinline__ void dma16x2(unsigned m0v, int v0, int v1, __amdgpu_buffer_rsrc_t srd, int soff) {
  asm volatile("s_mov_b32 m0, %0\n\ts_nop 0\n\t"
               "buffer_load_dwordx4 %1, %3, %4 offen lds\n\t"
               "buffer_load_dwordx4 %2, %3, %4 offen offset:1024 lds"
               :: "s"(m0v), "v"(v0), "v"(v1), "s"(srd), "s"(soff) : "memory");
}
__device__ __forceinline__ void dma16_sc(unsigned m0v, int voff, __amdgpu_buffer_rsrc_t srd, int soff) {
  asm volatile("s_mov_b32 m0, %0\n\ts_nop 0\n\tbuffer_load_dwordx4 %1, %2, %3 offen sc0 sc1 lds" :: "s"(m0v), "v"(voff), "s"(srd), "s"(soff) : "memory");
}
__device__ __forceinline__ void st16_sys(unsigned char* p, v4u v) {
#define PUSH_ST "sc0 sc1"
  asm volatile("global_store_dwordx4 %0, %1, off " PUSH_ST "\n\ts_nop 1" :: "v"(p), "v"(v) : "memory");
}
__device__ __forceinline__ void st4_sys(unsigned char* p, unsigned v) {
  asm volatile("global_store_dword %0, %1, off sc0 sc1" :: "v"(p), "v"(v) : "memory");
}
__device__ __forceinline__ v4u ld16_asm(const unsigned char* p) {
  v4u v;
  asm volatile("global_load_dwordx4 %0, %1, off" : "=v"(v) : "v"(p) : "memory");
  return v;
}
__device__ __forceinline__ int rfl(int v) { return __builtin_amdgcn_readfirstlane(v); }
__device__ __forceinline__ int lds_ld(LAS int* p) { return *(volatile LAS int*)p; }
__device__ __forceinline__ void lds_st(LAS int* p, int v) { *(volatile LAS int*)p = v; }

__global__ void __launch_bounds__(NTHREADS) fused_ag_gemm(Params p) {
  extern __shared__ __attribute__((aligned(16))) unsigned char smem[];
  LAS unsigned char* lds = (LAS unsigned char*)smem;
  LAS int* sync = (LAS int*)(lds + SYNC_OFF);
  const int tid = threadIdx.x, lane = tid & 63;
  const int wid = rfl(tid >> 6);
  const int bid = blockIdx.x;
  unsigned char* const myws = p.myws;
  const int srcr = bid & 7, idx = bid >> 3;
  const int rb = idx & 1, tn = idx >> 1;
  const int m0 = (srcr * 2 + rb) * BM, n0 = tn * BN;
  const bool remote = srcr != p.rank;
  const int q = !PUSH0 ? bid : bid >= 64 ? bid - 56 : bid >= 8 ? bid + 192 : -1;
  const bool pub = wid < 8 && q >= 0, psh = PUSH0 && wid == 10 && bid < 64;
  unsigned* const cnt = (unsigned*)(myws + CNT_OFF + (size_t)bid * 128);
  const unsigned prev = __hip_atomic_load(cnt, __ATOMIC_RELAXED, __HIP_MEMORY_SCOPE_AGENT);
  unsigned char* fp = p.ws[lane & (NR - 1)];
  v4u piece = {0u, 0u, 0u, 0u};
  if (pub) piece = *(const v4u*)(p.x + (size_t)((q & 7) * 64 + wid * 8 + (lane >> 3)) * K + (q >> 3) * BK + (lane & 7) * 16);
  if (psh) piece = *(const v4u*)(p.x + (size_t)(bid * 8 + (lane >> 3)) * K + (lane & 7) * 16);
  const unsigned ldsa = (unsigned)(size_t)lds;
  if (tid < 32) sync[tid] = (PUSH0 && tid == 14) ? 1 : 0;
  const unsigned call = (unsigned)rfl((int)prev) + 1u;
  asm volatile("" : "+v"(fp));
  if (pub) {
    st16_sys(myws + (size_t)(call & 1u) * SHARD_B + (size_t)(q >> 3) * STEP_B + ((q & 7) * 64 + wid * 8) * BK + lane * 16, piece);
    WAITV(0);
  } else if (psh) {
    const size_t doff = INBOX_OFF + (size_t)((call & 1u) * NR + p.rank) * STEP_B + bid * 1024 + lane * 16;
#pragma unroll
    for (int j = 0; j < NR - 1; ++j) st16_sys(p.peer[j] + doff, piece);
  } else if (wid == 11 && idx == 0) {
    __builtin_amdgcn_fence(__ATOMIC_ACQUIRE, "");
  }
  WAITL(0);
  __builtin_amdgcn_s_barrier();
  if (wid == 7) {
    if (q >= 0 && lane < NR && lane != p.rank) st4_sys(fp + FLAG_OFF + p.rank * FLAG_B + q * 4, call);
    if (lane == 0) __hip_atomic_store(cnt, call, __ATOMIC_RELAXED, __HIP_MEMORY_SCOPE_AGENT);
  }
  if (psh) {
    WAITV(0);
    if (lane < NR && lane != p.rank) st4_sys(fp + FLAG_OFF + p.rank * FLAG_B + 1024 + bid * 4, call);
  }
  const unsigned char* const xsrc = p.ws[srcr] + (size_t)(call & 1u) * SHARD_B;
  if (wid == 11) {
    const unsigned xcc = __builtin_amdgcn_s_getreg(20 | (3 << 11)) & 15u;
    unsigned* const invw = (unsigned*)(myws + INV_OFF + xcc * 128);
    if (idx == 0) {
      WAITV(0);
      if (lane == 0) __hip_atomic_store(invw, call, __ATOMIC_RELAXED, __HIP_MEMORY_SCOPE_AGENT);
    }
    if (!remote) return;
    const unsigned long long* fl = (const unsigned long long*)(myws + FLAG_OFF + srcr * FLAG_B) + lane * 2;
    const unsigned long long* fl2 = (const unsigned long long*)(myws + FLAG_OFF + srcr * FLAG_B + 1024) + (lane & 15) * 2;
    unsigned long long okp = 0ull;
    unsigned* const tmo = (unsigned*)(myws + TMO_OFF);
    const unsigned long long t0 = __builtin_amdgcn_s_memrealtime();
    bool inv_ok = idx == 0;
    for (unsigned it = 0;; ++it) {
      const unsigned long long f0 = __hip_atomic_load(fl, __ATOMIC_RELAXED, __HIP_MEMORY_SCOPE_SYSTEM);
      const unsigned long long f1 = __hip_atomic_load(fl + 1, __ATOMIC_RELAXED, __HIP_MEMORY_SCOPE_SYSTEM);
      unsigned long long g0 = 0ull, g1 = 0ull;
      if (PUSH0 && okp != 3ull) {
        g0 = __hip_atomic_load(fl2, __ATOMIC_RELAXED, __HIP_MEMORY_SCOPE_SYSTEM);
        g1 = __hip_atomic_load(fl2 + 1, __ATOMIC_RELAXED, __HIP_MEMORY_SCOPE_SYSTEM);
      }
      const unsigned iv = inv_ok ? call : __hip_atomic_load(invw, __ATOMIC_RELAXED, __HIP_MEMORY_SCOPE_AGENT);
      const bool ok = (int)((unsigned)f0 - call) >= 0 && (int)((unsigned)(f0 >> 32) - call) >= 0 &&
                      (int)((unsigned)f1 - call) >= 0 && (int)((unsigned)(f1 >> 32) - call) >= 0;
      unsigned long long okm = __ballot(ok);
      if (PUSH0) {
        if (okp != 3ull) {
          const bool ok2 = (int)((unsigned)g0 - call) >= 0 && (int)((unsigned)(g0 >> 32) - call) >= 0 &&
                           (int)((unsigned)g1 - call) >= 0 && (int)((unsigned)(g1 >> 32) - call) >= 0;
          const unsigned m2 = (unsigned)__ballot(ok2);
          okp = ((m2 & 0xffu) == 0xffu ? 1ull : 0ull) | ((m2 & 0xff00u) == 0xff00u ? 2ull : 0ull);
        }
        okm = (okm & ~3ull) | okp;
      }
      inv_ok = (int)((unsigned)rfl((int)iv) - call) >= 0;
      if (!inv_ok && okm != 0ull && __builtin_amdgcn_s_memrealtime() - t0 > 500ull) {
        __builtin_amdgcn_fence(__ATOMIC_ACQUIRE, "");
        WAITV(0);
        inv_ok = true;
      }
      if ((it & 15u) == 15u &&
          (rfl((int)__hip_atomic_load(tmo, __ATOMIC_RELAXED, __HIP_MEMORY_SCOPE_AGENT)) == (int)call ||
           __builtin_amdgcn_s_memrealtime() - t0 > DEADLINE_TICKS)) {
        if (lane == 0) { __hip_atomic_store(tmo, call, __ATOMIC_RELAXED, __HIP_MEMORY_SCOPE_AGENT); lds_st(sync + 3, 1); }
        okm = ~0ull; inv_ok = true;
      }
      if (inv_ok) {
        if (lane == 0) { lds_st(sync + 12, (int)(unsigned)okm); lds_st(sync + 13, (int)(unsigned)(okm >> 32)); }
        if (okm == ~0ull) break;
      }
    }
    WAITV(0);
    return;
  }
  if (wid >= 8) {
    const int b = wid - 8;
    const int rstride = remote ? BK : K, sstride = remote ? STEP_B : BK;
    const unsigned char* xb = remote ? xsrc + (size_t)rb * BM * BK : p.x + (size_t)rb * BM * K;
    auto rx = __builtin_amdgcn_make_buffer_rsrc((void*)xb, 0, (int)SHARD_B - rb * BM * rstride, 0x00020000);
    auto rw = __builtin_amdgcn_make_buffer_rsrc((void*)(p.w + p.rank * NS + n0), 0, K * NT - (p.rank * NS + n0), 0x00020000);
    const int lr = lane >> 3, lc = lane & 7;
    const int sw0 = (lc ^ (lr >> 1)) << 4, sw1 = (lc ^ (4 + (lr >> 1))) << 4;
    const int xv0 = lr * rstride + sw0, xv1 = (lr + 8) * rstride + sw1;
    const int wv0 = lr * NT + sw0, wv1 = lr * NT + sw1;
    const unsigned ldsb = ldsa + b * STAGE;
#define XM 2
#define WM 1
    const int xw1 = xv1 - 1024, xw2 = xv0 + 16 * rstride - 2048, xw3 = xv1 + 16 * rstride - 3072;
    const int ww1 = wv0 + 8 * NT - 1024, ww2 = wv1 + 16 * NT - 2048, ww3 = wv1 + 24 * NT - 3072;
    auto load_w = [&](int s) {
      if (WM == 1) {
#pragma unroll
        for (int qq = 0; qq < 4; ++qq) dma16x4(ldsb + XB + qq * 4096, wv0, ww1, ww2, ww3, rw, (s * BK + qq * 32) * NT);
        return;
      }
#pragma unroll 1
      for (int qq = 0; qq < 4; ++qq) {
        const unsigned lb = ldsb + XB + qq * 4096;
        const int wo = (s * BK + qq * 32) * NT;
        dma16(lb, wv0, rw, wo); dma16(lb + 1024, wv0, rw, wo + 8 * NT);
        dma16(lb + 2048, wv1, rw, wo + 16 * NT); dma16(lb + 3072, wv1, rw, wo + 24 * NT);
      }
    };
#define SCOUTM 1
    const bool scout = remote && SCOUTM >= 0 && (tn & SCOUTM) == 0;
    for (int s = b; s < NSTEP; s += NSTAGE) {
      if (s >= NSTAGE) {
        for (unsigned it = 0; it < LDS_SPIN_CAP; ++it) {
          const int d = lds_ld(sync + 4 + (lane & 7));
          if (__all(d >= s - 2)) break;
          __builtin_amdgcn_s_sleep(1);
        }
      }
#define XF 0
      const bool xfirst = remote && NSPLIT > 0 && s >= NSTEP - XF;
      if (!(scout && s < NSTAGE) && !xfirst) load_w(s);
      if (remote) {
        const int ub = 2 * s + rb;
        for (unsigned it = 0; it < LDS_SPIN_CAP; ++it) {
          const int v = lds_ld(sync + 12 + (lane & 3));
          const unsigned mw = (unsigned)__builtin_amdgcn_readlane(v, ub >> 5);
          if (((mw >> (ub & 31)) & 1u) && __builtin_amdgcn_readlane(v, 2) >= s) break;
          __builtin_amdgcn_s_sleep(1);
        }
#define SKEW 50
        if (rb && s == 0) {
          const unsigned long long t1 = __builtin_amdgcn_s_memrealtime();
          while (__builtin_amdgcn_s_memrealtime() - t1 < SKEW) __builtin_amdgcn_s_sleep(1);
        }
      }
      const int so = s * sstride;
      if (PUSH0 && remote && s == 0) {
        auto ri = __builtin_amdgcn_make_buffer_rsrc((void*)(myws + INBOX_OFF + (size_t)((call & 1u) * NR + srcr) * STEP_B + rb * XB), 0, XB, 0x00020000);
#pragma unroll 1
        for (int pp = 0; pp < 16; ++pp) {
          dma16_sc(ldsb + pp * 2048, xv0, ri, pp * 2048);
          dma16_sc(ldsb + pp * 2048 + 1024, xv1, ri, pp * 2048);
        }
        WAITV(0);
        if (lane < 2) lds_st(sync + lane * 16 + b, s + 1);
      } else if (remote && s >= NSTEP - NSPLIT) {
#pragma unroll 1
        for (int pp = 0; pp < 8; ++pp) {
          const int pr = (pp + tn) & 7;
          dma16(ldsb + pr * 2048, xv0, rx, so + pr * 16 * rstride);
          dma16(ldsb + pr * 2048 + 1024, xv1, rx, so + pr * 16 * rstride);
        }
        if (lane == 0) lds_st(sync + 14, s + 1);
        if (xfirst) load_w(s);
        WAITV(0);
        if (lane == 0) lds_st(sync + b, s + 1);
        for (unsigned it = 0; it < LDS_SPIN_CAP; ++it) { if (rfl(lds_ld(sync + 14)) >= s + NSPLIT) break; __builtin_amdgcn_s_sleep(1); }
#pragma unroll 1
        for (int pp = 0; pp < 8; ++pp) {
          const int pr = 8 + ((pp + tn) & 7);
          dma16(ldsb + pr * 2048, xv0, rx, so + pr * 16 * rstride);
          dma16(ldsb + pr * 2048 + 1024, xv1, rx, so + pr * 16 * rstride);
        }
        if (lane == 0) lds_st(sync + 14, s + NSPLIT + 1);
        WAITV(0);
        if (lane == 0) lds_st(sync + 16 + b, s + 1);
      } else {
#pragma unroll 2
        for (int pp = 0; pp < 16; ++pp) {
          const int pr = (pp + tn) & 15;
          dma16x2(ldsb + pr * 2048, xv0, xw1, rx, so + pr * 16 * rstride);
        }
        if (lane == 0) lds_st(sync + 14, s + 1);
        if (scout && s < NSTAGE) load_w(s);
        WAITV(0);
        if (lane < 2) lds_st(sync + lane * 16 + b, s + 1);
      }
    }
    return;
  }
  const int half = wid >> 2, wm = half * 2 + (wid & 1), wn = (wid >> 1) & 1;
  LAS int* const fullw = sync + half * 16;
  const int r = lane & 15, g = lane >> 4;
  const int xa0 = (wm * 64 + r) * 128 + ((g ^ ((r >> 1) & 7)) << 4);
  const int xa1 = xa0 ^ 64;
  const int fw = ((r >> 2) & 3) | ((g & 1) << 2);
  const int wa = XB + (16 * g + (r >> 1)) * 128 + (r & 1) * 8;
  int wan[4];
#pragma unroll
  for (int nt = 0; nt < 4; ++nt) wan[nt] = wa + (((wn * 4 + nt) ^ fw) << 4);
  v4f acc[4][4];
#pragma unroll
  for (int i = 0; i < 4; ++i)
#pragma unroll
    for (int j = 0; j < 4; ++j) acc[i][j] = (v4f){0.f, 0.f, 0.f, 0.f};
  float sc = p.sx[0] * p.sw[0];
  asm volatile("" : "+v"(sc));
  int buf = 0;
  int fullv = 0;
  for (int s = 0; s < NSTEP; ++s) {
    if (rfl(fullv) <= s) {
      for (unsigned it = 0; it < LDS_SPIN_CAP; ++it) {
        fullv = lds_ld(fullw + buf);
        if (rfl(fullv) > s) break;
        __builtin_amdgcn_s_sleep(1);
      }
    }
    const int sb = buf * STAGE;
    v8i xf[4], wf[4];
#pragma unroll
    for (int nt = 0; nt < 4; ++nt) {
      v2i q[4];
#pragma unroll
      for (int hs = 0; hs < 4; ++hs)
        q[hs] = __builtin_amdgcn_ds_read_tr8_b64_v2i32((LAS v2i*)(lds + sb + wan[nt] + (hs >> 1) * 8192 + (hs & 1) * 1024));
      wf[nt] = (v8i){q[0].x, q[0].y, q[1].x, q[1].y, q[2].x, q[2].y, q[3].x, q[3].y};
    }
#pragma unroll
    for (int mt = 0; mt < 4; ++mt) {
      v4i a = *(LAS v4i*)(lds + sb + xa0 + mt * 2048), c = *(LAS v4i*)(lds + sb + xa1 + mt * 2048);
      xf[mt] = (v8i){a.x, a.y, a.z, a.w, c.x, c.y, c.z, c.w};
    }
    buf = buf + 1; if (buf >= NSTAGE) buf = 0;
    __builtin_amdgcn_s_waitcnt(0xC07F);
    __builtin_amdgcn_sched_barrier(0);
    lds_st(sync + 4 + wid, s + 1);
    fullv = lds_ld(fullw + buf);
    __builtin_amdgcn_sched_barrier(0);
    __builtin_amdgcn_s_setprio(1);
#pragma unroll
    for (int mt = 0; mt < 4; ++mt)
#pragma unroll
      for (int nt = 0; nt < 4; ++nt)
        acc[mt][nt] = __builtin_amdgcn_mfma_scale_f32_16x16x128_f8f6f4(wf[nt], xf[mt], acc[mt][nt], 1, 0, 0, 0, 0, 0);
    __builtin_amdgcn_s_setprio(0);
    __builtin_amdgcn_sched_barrier(0);
  }
  for (unsigned it = 0; it < LDS_SPIN_CAP; ++it) {
    const int d = lds_ld(sync + 4 + half * 4 + (lane & 3));
    if (__all(d >= NSTEP - 2)) break;
    __builtin_amdgcn_s_sleep(1);
  }
  const bool bad = rfl(lds_ld(sync + 3)) != 0;
  LAS unsigned char* eb = lds + ((NSTEP - 3) % NSTAGE) * STAGE + wid * 4096;
  float* o = p.out + (size_t)(m0 + wm * 64 + g) * NS + n0 + wn * 64 + r * 4;
#pragma unroll
  for (int mt = 0; mt < 4; ++mt) {
#pragma unroll
    for (int nt = 0; nt < 4; ++nt) {
      v4f v = acc[mt][nt] * sc;
      v.x = fmaxf(v.x, 0.f); v.y = fmaxf(v.y, 0.f); v.z = fmaxf(v.z, 0.f); v.w = fmaxf(v.w, 0.f);
      if (bad) { const float q = __builtin_nanf(""); v = (v4f){q, q, q, q}; }
      *(LAS v4f*)(eb + r * 256 + (((nt * 4 + g) ^ r) << 4)) = v;
    }
#pragma unroll
    for (int i = 0; i < 4; ++i) {
      const v4f v = *(LAS v4f*)(eb + (i * 4 + g) * 256 + ((r ^ (i * 4 + g)) << 4));
      asm volatile("global_store_dwordx4 %0, %1, off " OUT_ST "\n\ts_nop 1" :: "v"(o + (size_t)(mt * 16 + i * 4) * NS), "v"(v) : "memory");
    }
  }
}
}

extern "C" void kernel_launch(int rank, int nranks, ncclComm_t comm,
                              hipStream_t stream,
                              void* const* d_in, const int* in_sizes, int n_in,
                              void* d_out, int out_size,
                              const int* in_dtypes, int out_dtype,
                              void* const* d_out_all,
                              void* const* d_ws_all, size_t ws_bytes) {
  (void)comm; (void)in_sizes; (void)n_in; (void)out_size; (void)in_dtypes; (void)out_dtype; (void)d_out_all;
  if (nranks != NR || (unsigned)rank >= (unsigned)NR || ws_bytes < WS_NEED) {
    fprintf(stderr, "unexpected configuration\n");
    return;
  }
  Params p;
  p.x = (const unsigned char*)d_in[0];
  p.w = (const unsigned char*)d_in[1];
  p.sx = (const float*)d_in[2];
  p.sw = (const float*)d_in[3];
  p.out = (float*)d_out;
  for (int i = 0; i < NR; ++i) p.ws[i] = (unsigned char*)d_ws_all[i];
  p.myws = p.ws[rank];
  for (int i = 0; i < NR - 1; ++i) p.peer[i] = p.ws[(rank + 1 + i) % NR];
  p.rank = rank;
  HIP_CHECK(hipFuncSetAttribute((const void*)fused_ag_gemm, hipFuncAttributeMaxDynamicSharedMemorySize, LDS_BYTES));
  fused_ag_gemm<<<dim3(NWG), dim3(NTHREADS), LDS_BYTES, stream>>>(p);
  HIP_CHECK(hipGetLastError());
}
```

```cpp
#include <hip/hip_runtime.h>
#include <rccl/rccl.h>
#include <cstdio>
#include <cstdint>

#define HIP_CHECK(expr) do { hipError_t _e = (expr); if (_e != hipSuccess) { \
    fprintf(stderr, "HIP error %s at %s:%d\n", hipGetErrorString(_e), __FILE__, __LINE__); \
    return; } } while (0)

#define OUT_ST "nt"
namespace {
typedef int v8i __attribute__((ext_vector_type(8)));
typedef int v4i __attribute__((ext_vector_type(4)));
typedef int v2i __attribute__((ext_vector_type(2)));
typedef float v4f __attribute__((ext_vector_type(4)));
typedef unsigned v4u __attribute__((ext_vector_type(4)));
#define LAS __attribute__((address_space(3)))

constexpr int NR = 8;
constexpr int M = 4096, K = 4096, NT = 16384, NS = NT / NR, MS = M / NR;
constexpr int BM = 256, BN = 128, BK = 128, NSTEP = K / BK;
constexpr int XB = BM * BK, WB = BK * BN, STAGE = XB + WB, NSTAGE = 3;
constexpr int SYNC_OFF = STAGE * NSTAGE;
constexpr int LDS_BYTES = SYNC_OFF + 128;
constexpr int NTHREADS = 768, NWG = 256;
constexpr int STEP_B = MS * BK;
constexpr size_t SHARD_B = (size_t)MS * K;
constexpr size_t FLAG_OFF = 2 * SHARD_B;
constexpr int FLAG_B = 1024;
constexpr size_t CNT_OFF = FLAG_OFF + NR * FLAG_B;
constexpr size_t INV_OFF = CNT_OFF + (size_t)NWG * 128;
constexpr size_t TMO_OFF = INV_OFF + 16 * 128;
constexpr size_t WS_NEED = TMO_OFF + 4096;
#define INVIDX 12
#define SKEW 50
#define NSPLIT 3
constexpr unsigned long long DEADLINE_TICKS = 50000000ull;
constexpr unsigned LDS_SPIN_CAP = 1u << 26;

struct Params {
  const unsigned char* x;
  unsigned char* myws;
  const unsigned char* w;
  float* out;
  const float* sx;
  const float* sw;
  int rank, pad;
  unsigned char* ws[NR];
};

#define WAITV_(n) asm volatile("s_waitcnt vmcnt(" #n ")" ::: "memory")
#define WAITV(n) WAITV_(n)
#define WAITL(n) asm volatile("s_waitcnt lgkmcnt(" #n ")" ::: "memory")

__device__ __forceinline__ void dma16x4(unsigned m0v, int v0, int v1, int v2, int v3, __amdgpu_buffer_rsrc_t srd, int soff) {
  asm volatile("s_mov_b32 m0, %0\n\ts_nop 0\n\t"
               "buffer_load_dwordx4 %1, %5, %6 offen lds\n\t"
               "buffer_load_dwordx4 %2, %5, %6 offen offset:1024 lds\n\t"
               "buffer_load_dwordx4 %3, %5, %6 offen offset:2048 lds\n\t"
               "buffer_load_dwordx4 %4, %5, %6 offen offset:3072 lds"
               :: "s"(m0v), "v"(v0), "v"(v1), "v"(v2), "v"(v3), "s"(srd), "s"(soff) : "memory");
}
__device__ __forceinline__ void dma16x2(unsigned m0v, int v0, int v1, __amdgpu_buffer_rsrc_t srd, int soff) {
  asm volatile("s_mov_b32 m0, %0\n\ts_nop 0\n\t"
               "buffer_load_dwordx4 %1, %3, %4 offen lds\n\t"
               "buffer_load_dwordx4 %2, %3, %4 offen offset:1024 lds"
               :: "s"(m0v), "v"(v0), "v"(v1), "s"(srd), "s"(soff) : "memory");
}
__device__ __forceinline__ void st16_sys(unsigned char* p, v4u v) {
  asm volatile("global_store_dwordx4 %0, %1, off sc0 sc1\n\ts_nop 1" :: "v"(p), "v"(v) : "memory");
}
__device__ __forceinline__ void st4_sys(unsigned char* p, unsigned v) {
  asm volatile("global_store_dword %0, %1, off sc0 sc1" :: "v"(p), "v"(v) : "memory");
}
__device__ __forceinline__ int rfl(int v) { return __builtin_amdgcn_readfirstlane(v); }
__device__ __forceinline__ int lds_ld(LAS int* p) { return *(volatile LAS int*)p; }
__device__ __forceinline__ void lds_st(LAS int* p, int v) { *(volatile LAS int*)p = v; }

__global__ void __launch_bounds__(NTHREADS) fused_ag_gemm(Params p) {
  extern __shared__ __attribute__((aligned(16))) unsigned char smem[];
  LAS unsigned char* lds = (LAS unsigned char*)smem;
  LAS int* sync = (LAS int*)(lds + SYNC_OFF);
  const int tid = threadIdx.x, lane = tid & 63;
  const int wid = rfl(tid >> 6);
  const int bid = blockIdx.x;
  unsigned char* myws = p.myws;
  const unsigned char* px = p.x;
  int rank = p.rank;
  asm volatile("" : "+s"(rank));
  const int srcr = bid & 7, idx = bid >> 3;
  const int rb = idx & 1, tn = idx >> 1;
  const int m0 = (srcr * 2 + rb) * BM, n0 = tn * BN;
  const bool remote = srcr != rank;
  unsigned* const cnt = (unsigned*)(myws + CNT_OFF + (size_t)bid * 128);
  const unsigned prev = __hip_atomic_load(cnt, __ATOMIC_RELAXED, __HIP_MEMORY_SCOPE_AGENT);
  v4u piece = {0u, 0u, 0u, 0u};
  if (wid < 8) piece = *(const v4u*)(px + (size_t)((bid & 7) * 64 + wid * 8 + (lane >> 3)) * K + (bid >> 3) * BK + (lane & 7) * 16);
  unsigned long long fp = 0;
  if (wid == 7) {
#pragma unroll
    for (int i = 0; i < NR; ++i) {
      unsigned long long w = (unsigned long long)p.ws[i];
      asm volatile("" : "+s"(w));
      fp = (lane == i) ? w : fp;
    }
  }
  const unsigned ldsa = (unsigned)(size_t)lds;
  const unsigned xcc = __builtin_amdgcn_s_getreg(20 | (3 << 11)) & 15u;
  unsigned* const invw = (unsigned*)(myws + INV_OFF + xcc * 128);
  if (tid < 32) sync[tid] = 0;
  const unsigned call = (unsigned)rfl((int)prev) + 1u;
  asm volatile("" :: "s"(call) : "memory");
  if (wid < 8) {
    st16_sys(myws + (size_t)(call & 1u) * SHARD_B + (size_t)(bid >> 3) * STEP_B + ((bid & 7) * 64 + wid * 8) * BK + lane * 16, piece);
    WAITV(0);
  } else if (wid == 11 && idx == INVIDX) {
    __builtin_amdgcn_fence(__ATOMIC_ACQUIRE, "");
    WAITV(0);
    if (lane == 0) __hip_atomic_store(invw, call, __ATOMIC_RELAXED, __HIP_MEMORY_SCOPE_AGENT);
  }
  WAITL(0);
  __builtin_amdgcn_s_barrier();
  if (wid == 7) {
    if (lane < NR && lane != rank) st4_sys((unsigned char*)fp + FLAG_OFF + rank * FLAG_B + bid * 4, call);
    if (lane == 0) __hip_atomic_store(cnt, call, __ATOMIC_RELAXED, __HIP_MEMORY_SCOPE_AGENT);
  }
  const unsigned char* const xsrc = p.ws[srcr] + (size_t)(call & 1u) * SHARD_B;
  if (wid == 11) {
    if (!remote) return;
    const unsigned long long* fl = (const unsigned long long*)(myws + FLAG_OFF + srcr * FLAG_B) + lane * 2;
    unsigned* const tmo = (unsigned*)(myws + TMO_OFF);
    const unsigned long long t0 = __builtin_amdgcn_s_memrealtime();
    bool inv_ok = idx == INVIDX;
    for (unsigned it = 0;; ++it) {
      const unsigned long long f0 = __hip_atomic_load(fl, __ATOMIC_RELAXED, __HIP_MEMORY_SCOPE_SYSTEM);
      const unsigned long long f1 = __hip_atomic_load(fl + 1, __ATOMIC_RELAXED, __HIP_MEMORY_SCOPE_SYSTEM);
      const unsigned iv = inv_ok ? call : __hip_atomic_load(invw, __ATOMIC_RELAXED, __HIP_MEMORY_SCOPE_AGENT);
      const bool ok = (int)((unsigned)f0 - call) >= 0 && (int)((unsigned)(f0 >> 32) - call) >= 0 &&
                      (int)((unsigned)f1 - call) >= 0 && (int)((unsigned)(f1 >> 32) - call) >= 0;
      unsigned long long okm = __ballot(ok);
      inv_ok = (int)((unsigned)rfl((int)iv) - call) >= 0;
      if (!inv_ok && okm != 0ull && __builtin_amdgcn_s_memrealtime() - t0 > 500ull) {
        __builtin_amdgcn_fence(__ATOMIC_ACQUIRE, "");
        WAITV(0);
        inv_ok = true;
      }
      if ((it & 15u) == 15u &&
          (rfl((int)__hip_atomic_load(tmo, __ATOMIC_RELAXED, __HIP_MEMORY_SCOPE_AGENT)) == (int)call ||
           __builtin_amdgcn_s_memrealtime() - t0 > DEADLINE_TICKS)) {
        if (lane == 0) { __hip_atomic_store(tmo, call, __ATOMIC_RELAXED, __HIP_MEMORY_SCOPE_AGENT); lds_st(sync + 3, 1); }
        okm = ~0ull; inv_ok = true;
      }
      if (inv_ok) {
        if (lane == 0) { lds_st(sync + 12, (int)(unsigned)okm); lds_st(sync + 13, (int)(unsigned)(okm >> 32)); }
        if (okm == ~0ull) break;
      }
    }
    WAITV(0);
    return;
  }
  if (wid >= 8) {
    const int b = wid - 8;
    const int rstride = remote ? BK : K, sstride = remote ? STEP_B : BK;
    const unsigned char* xb = remote ? xsrc + (size_t)rb * BM * BK : px + (size_t)rb * BM * K;
    auto rx = __builtin_amdgcn_make_buffer_rsrc((void*)xb, 0, (int)SHARD_B - rb * BM * rstride, 0x00020000);
    auto rw = __builtin_amdgcn_make_buffer_rsrc((void*)(p.w + rank * NS + n0), 0, K * NT - (rank * NS + n0), 0x00020000);
    const int lr = lane >> 3, lc = lane & 7;
    const int sw0 = (lc ^ (lr >> 1)) << 4, sw1 = (lc ^ (4 + (lr >> 1))) << 4;
    const int xv0 = lr * rstride + sw0, xv1 = (lr + 8) * rstride + sw1 - 1024;
    const int wv0 = lr * NT + sw0, wv1 = (lr + 8) * NT + sw0 - 1024, wv2 = (lr + 16) * NT + sw1 - 2048, wv3 = (lr + 24) * NT + sw1 - 3072;
    const unsigned ldsb = ldsa + b * STAGE;
    const bool scout = remote && (tn & 1) == 0;
    for (int s = b; s < NSTEP; s += NSTAGE) {
      if (s >= NSTAGE) {
        for (unsigned it = 0; it < LDS_SPIN_CAP; ++it) {
          const int d = lds_ld(sync + 4 + (lane & 7));
          if (__all(d >= s - 2)) break;
          __builtin_amdgcn_s_sleep(1);
        }
      }
      const bool split = remote && s >= NSTEP - NSPLIT;
      const bool wlate = scout && s < NSTAGE;
      const int np = split ? 8 : 16;
#pragma unroll 1
      for (int job = wlate ? 1 : 0; job < (split ? 4 : 3); ++job) {
        if (job == 0 || (job == 2 && wlate)) {
#pragma unroll
          for (int qq = 0; qq < 4; ++qq) dma16x4(ldsb + XB + qq * 4096, wv0, wv1, wv2, wv3, rw, (s * BK + qq * 32) * NT);
        }
        if (job & 1) {
          const int turn = job == 1 ? s : s + NSPLIT;
          if (remote) {
            const int ub = 2 * s + rb;
            for (unsigned it = 0; it < LDS_SPIN_CAP; ++it) {
              const int v = lds_ld(sync + 12 + (lane & 3));
              const unsigned mw = (unsigned)__builtin_amdgcn_readlane(v, ub >> 5);
              if (((mw >> (ub & 31)) & 1u) && __builtin_amdgcn_readlane(v, 2) >= turn) break;
              __builtin_amdgcn_s_sleep(1);
            }
            if (SKEW > 0 && rb && s == 0) {
              const unsigned long long t1 = __builtin_amdgcn_s_memrealtime();
              while (__builtin_amdgcn_s_memrealtime() - t1 < SKEW) __builtin_amdgcn_s_sleep(1);
            }
          }
          const int pb = job == 1 ? 0 : 8;
#pragma unroll 2
          for (int pp = 0; pp < np; ++pp) {
            const int pr = pb + ((pp + tn) & (np - 1));
            dma16x2(ldsb + pr * 2048, xv0, xv1, rx, s * sstride + pr * 16 * rstride);
          }
          if (lane == 0) lds_st(sync + 14, turn + 1);
        }
        if (job >= 2) {
          WAITV(0);
          if (job == 2 ? (lane == 0 || (lane == 1 && !split)) : lane == 1) lds_st(sync + lane * 16 + b, s + 1);
        }
      }
    }
    return;
  }
  const int half = wid >> 2, wm = half * 2 + (wid & 1), wn = (wid >> 1) & 1;
  LAS int* const fullw = sync + half * 16;
  const int r = lane & 15, g = lane >> 4;
  const int xa0 = (wm * 64 + r) * 128 + ((g ^ ((r >> 1) & 7)) << 4);
  const int xa1 = xa0 ^ 64;
  const int fw = ((r >> 2) & 3) | ((g & 1) << 2);
  const int wa = XB + (16 * g + (r >> 1)) * 128 + (r & 1) * 8;
  int wan[4];
#pragma unroll
  for (int nt = 0; nt < 4; ++nt) wan[nt] = wa + (((wn * 4 + nt) ^ fw) << 4);
  v4f acc[4][4];
#pragma unroll
  for (int i = 0; i < 4; ++i)
#pragma unroll
    for (int j = 0; j < 4; ++j) acc[i][j] = (v4f){0.f, 0.f, 0.f, 0.f};
  float sc = p.sx[0] * p.sw[0];
  asm volatile("" : "+v"(sc));
  int buf = 0;
  int fullv = 0;
  for (int s = 0; s < NSTEP; ++s) {
    if (rfl(fullv) <= s) {
      for (unsigned it = 0; it < LDS_SPIN_CAP; ++it) {
        fullv = lds_ld(fullw + buf);
        if (rfl(fullv) > s) break;
        __builtin_amdgcn_s_sleep(1);
      }
    }
    const int sb = buf * STAGE;
    v8i xf[4], wf[4];
#pragma unroll
    for (int nt = 0; nt < 4; ++nt) {
      v2i q[4];
#pragma unroll
      for (int hs = 0; hs < 4; ++hs)
        q[hs] = __builtin_amdgcn_ds_read_tr8_b64_v2i32((LAS v2i*)(lds + sb + wan[nt] + (hs >> 1) * 8192 + (hs & 1) * 1024));
      wf[nt] = (v8i){q[0].x, q[0].y, q[1].x, q[1].y, q[2].x, q[2].y, q[3].x, q[3].y};
    }
#pragma unroll
    for (int mt = 0; mt < 4; ++mt) {
      v4i a = *(LAS v4i*)(lds + sb + xa0 + mt * 2048), c = *(LAS v4i*)(lds + sb + xa1 + mt * 2048);
      xf[mt] = (v8i){a.x, a.y, a.z, a.w, c.x, c.y, c.z, c.w};
    }
    buf = buf + 1; if (buf >= NSTAGE) buf = 0;
    __builtin_amdgcn_s_waitcnt(0xC07F);
    __builtin_amdgcn_sched_barrier(0);
    lds_st(sync + 4 + wid, s + 1);
    fullv = lds_ld(fullw + buf);
    __builtin_amdgcn_sched_barrier(0);
    __builtin_amdgcn_s_setprio(1);
#pragma unroll
    for (int mt = 0; mt < 4; ++mt)
#pragma unroll
      for (int nt = 0; nt < 4; ++nt)
        acc[mt][nt] = __builtin_amdgcn_mfma_scale_f32_16x16x128_f8f6f4(wf[nt], xf[mt], acc[mt][nt], 1, 0, 0, 0, 0, 0);
    __builtin_amdgcn_s_setprio(0);
    __builtin_amdgcn_sched_barrier(0);
  }
  for (unsigned it = 0; it < LDS_SPIN_CAP; ++it) {
    const int d = lds_ld(sync + 4 + half * 4 + (lane & 3));
    if (__all(d >= NSTEP - 2)) break;
    __builtin_amdgcn_s_sleep(1);
  }
  const bool bad = rfl(lds_ld(sync + 3)) != 0;
  LAS unsigned char* eb = lds + ((NSTEP - 3) % NSTAGE) * STAGE + wid * 4096;
  float* o = p.out + (size_t)(m0 + wm * 64 + g) * NS + n0 + wn * 64 + r * 4;
#pragma unroll
  for (int mt = 0; mt < 4; ++mt) {
#pragma unroll
    for (int nt = 0; nt < 4; ++nt) {
      v4f v = acc[mt][nt] * sc;
      v.x = fmaxf(v.x, 0.f); v.y = fmaxf(v.y, 0.f); v.z = fmaxf(v.z, 0.f); v.w = fmaxf(v.w, 0.f);
      if (bad) { const float q = __builtin_nanf(""); v = (v4f){q, q, q, q}; }
      *(LAS v4f*)(eb + r * 256 + (((nt * 4 + g) ^ r) << 4)) = v;
    }
#pragma unroll
    for (int i = 0; i < 4; ++i) {
      const v4f v = *(LAS v4f*)(eb + (i * 4 + g) * 256 + ((r ^ (i * 4 + g)) << 4));
      asm volatile("global_store_dwordx4 %0, %1, off " OUT_ST "\n\ts_nop 1" :: "v"(o + (size_t)(mt * 16 + i * 4) * NS), "v"(v) : "memory");
    }
  }
}
}

extern "C" void kernel_launch(int rank, int nranks, ncclComm_t comm,
                              hipStream_t stream,
                              void* const* d_in, const int* in_sizes, int n_in,
                              void* d_out, int out_size,
                              const int* in_dtypes, int out_dtype,
                              void* const* d_out_all,
                              void* const* d_ws_all, size_t ws_bytes) {
  (void)comm; (void)in_sizes; (void)n_in; (void)out_size; (void)in_dtypes; (void)out_dtype; (void)d_out_all;
  if (nranks != NR || (unsigned)rank >= (unsigned)NR || ws_bytes < WS_NEED) {
    fprintf(stderr, "unexpected configuration\n");
    return;
  }
  Params p;
  p.x = (const unsigned char*)d_in[0];
  p.w = (const unsigned char*)d_in[1];
  p.sx = (const float*)d_in[2];
  p.sw = (const float*)d_in[3];
  p.out = (float*)d_out;
  for (int i = 0; i < NR; ++i) p.ws[i] = (unsigned char*)d_ws_all[i];
  p.myws = p.ws[rank];
  p.rank = rank;
  p.pad = 0;
  HIP_CHECK(hipFuncSetAttribute((const void*)fused_ag_gemm, hipFuncAttributeMaxDynamicSharedMemorySize, LDS_BYTES));
  fused_ag_gemm<<<dim3(NWG), dim3(NTHREADS), LDS_BYTES, stream>>>(p);
  HIP_CHECK(hipGetLastError());
}
```

```cpp
#include <hip/hip_runtime.h>
#include <rccl/rccl.h>
#include <cstdio>
#include <cstdint>

#define HIP_CHECK(expr) do { hipError_t _e = (expr); if (_e != hipSuccess) { \
    fprintf(stderr, "HIP error %s at %s:%d\n", hipGetErrorString(_e), __FILE__, __LINE__); \
    return; } } while (0)

namespace {
typedef int v8i __attribute__((ext_vector_type(8)));
typedef int v4i __attribute__((ext_vector_type(4)));
typedef int v2i __attribute__((ext_vector_type(2)));
typedef float v4f __attribute__((ext_vector_type(4)));
typedef unsigned v4u __attribute__((ext_vector_type(4)));
#define LAS __attribute__((address_space(3)))

constexpr int NR = 8;
constexpr int M = 4096, K = 4096, NT = 16384, NS = NT / NR, MS = M / NR;
constexpr int BM = 256, BN = 128, BK = 128, NSTEP = K / BK;
constexpr int XB = BM * BK, WB = BK * BN, STAGE = XB + WB, NSTAGE = 3;
constexpr int SYNC_OFF = STAGE * NSTAGE;
constexpr int LDS_BYTES = SYNC_OFF + 128;
constexpr int NTHREADS = 768, NWG = 256;
constexpr int STEP_B = MS * BK;
constexpr size_t SHARD_B = (size_t)MS * K;
constexpr size_t FLAG_OFF = 2 * SHARD_B;
constexpr int FLAG_B = 1024;
constexpr size_t CNT_OFF = FLAG_OFF + NR * FLAG_B;
constexpr size_t INV_OFF = CNT_OFF + (size_t)NWG * 128;
constexpr size_t TMO_OFF = INV_OFF + 16 * 128;
constexpr size_t WS_NEED = TMO_OFF + 4096;
#define INVIDX 12
#define SKEW 50
#define NSPLIT 3
constexpr unsigned long long DEADLINE_TICKS = 150000000ull;
constexpr unsigned LDS_SPIN_CAP = 1u << 26;

struct Params {
  const unsigned char* x;
  unsigned char* myws;
  const unsigned char* w;
  float* out;
  const float* sx;
  const float* sw;
  int rank, pad;
  unsigned char* ws[NR];
};

#define WAITV_(n) asm volatile("s_waitcnt vmcnt(" #n ")" ::: "memory")
#define WAITV(n) WAITV_(n)
#define WAITL(n) asm volatile("s_waitcnt lgkmcnt(" #n ")" ::: "memory")

__device__ __forceinline__ void dma16x4(unsigned m0v, int v0, int v1, int v2, int v3, __amdgpu_buffer_rsrc_t srd, int soff) {
  asm volatile("s_mov_b32 m0, %0\n\ts_nop 0\n\t"
               "buffer_load_dwordx4 %1, %5, %6 offen lds\n\t"
               "buffer_load_dwordx4 %2, %5, %6 offen offset:1024 lds\n\t"
               "buffer_load_dwordx4 %3, %5, %6 offen offset:2048 lds\n\t"
               "buffer_load_dwordx4 %4, %5, %6 offen offset:3072 lds"
               :: "s"(m0v), "v"(v0), "v"(v1), "v"(v2), "v"(v3), "s"(srd), "s"(soff) : "memory");
}
__device__ __forceinline__ void dma16x2(unsigned m0v, int v0, int v1, __amdgpu_buffer_rsrc_t srd, int soff) {
  asm volatile("s_mov_b32 m0, %0\n\ts_nop 0\n\t"
               "buffer_load_dwordx4 %1, %3, %4 offen lds\n\t"
               "buffer_load_dwordx4 %2, %3, %4 offen offset:1024 lds"
               :: "s"(m0v), "v"(v0), "v"(v1), "s"(srd), "s"(soff) : "memory");
}
__device__ __forceinline__ void st16_sys(unsigned char* p, v4u v) {
  asm volatile("global_store_dwordx4 %0, %1, off sc0 sc1\n\ts_nop 1" :: "v"(p), "v"(v) : "memory");
}
__device__ __forceinline__ void st4_sys(unsigned char* p, unsigned v) {
  asm volatile("global_store_dword %0, %1, off sc0 sc1" :: "v"(p), "v"(v) : "memory");
}
__device__ __forceinline__ int rfl(int v) { return __builtin_amdgcn_readfirstlane(v); }
__device__ __forceinline__ int lds_ld(LAS int* p) { return *(volatile LAS int*)p; }
__device__ __forceinline__ void lds_st(LAS int* p, int v) { *(volatile LAS int*)p = v; }

__global__ void __launch_bounds__(NTHREADS) fused_ag_gemm(Params p) {
  extern __shared__ __attribute__((aligned(16))) unsigned char smem[];
  LAS unsigned char* lds = (LAS unsigned char*)smem;
  LAS int* sync = (LAS int*)(lds + SYNC_OFF);
  const int tid = threadIdx.x, lane = tid & 63;
  const int wid = rfl(tid >> 6);
  const int bid = blockIdx.x;
  unsigned char* myws = p.myws;
  const unsigned char* px = p.x;
  int rank = p.rank;
  asm volatile("" : "+s"(rank));
  const int srcr = bid & 7, idx = bid >> 3;
  const int rb = idx & 1, tn = idx >> 1;
  const int m0 = (srcr * 2 + rb) * BM, n0 = tn * BN;
  const bool remote = srcr != rank;
  unsigned* const cnt = (unsigned*)(myws + CNT_OFF + (size_t)bid * 128);
  const unsigned prev = __hip_atomic_load(cnt, __ATOMIC_RELAXED, __HIP_MEMORY_SCOPE_AGENT);
  v4u piece = {0u, 0u, 0u, 0u};
  if (wid < 8) piece = *(const v4u*)(px + (size_t)((bid & 7) * 64 + wid * 8 + (lane >> 3)) * K + (bid >> 3) * BK + (lane & 7) * 16);
  unsigned long long fp = 0;
  if (wid == 7) {
#pragma unroll
    for (int i = 0; i < NR; ++i) {
      unsigned long long w = (unsigned long long)p.ws[i];
      asm volatile("" : "+s"(w));
      fp = (lane == i) ? w : fp;
    }
  }
  const unsigned ldsa = (unsigned)(size_t)lds;
  const unsigned xcc = __builtin_amdgcn_s_getreg(20 | (3 << 11)) & 15u;
  unsigned* const invw = (unsigned*)(myws + INV_OFF + xcc * 128);
  if (tid < 32) sync[tid] = 0;
  const unsigned call = (unsigned)rfl((int)prev) + 1u;
  asm volatile("" :: "s"(call) : "memory");
  if (wid < 8) {
    st16_sys(myws + (size_t)(call & 1u) * SHARD_B + (size_t)(bid >> 3) * STEP_B + ((bid & 7) * 64 + wid * 8) * BK + lane * 16, piece);
    WAITV(0);
  } else if (wid == 11 && idx == INVIDX) {
    __builtin_amdgcn_fence(__ATOMIC_ACQUIRE, "");
    WAITV(0);
    if (lane == 0) __hip_atomic_store(invw, call, __ATOMIC_RELAXED, __HIP_MEMORY_SCOPE_AGENT);
  }
  WAITL(0);
  __builtin_amdgcn_s_barrier();
  if (wid == 7) {
    if (lane < NR && lane != rank) st4_sys((unsigned char*)fp + FLAG_OFF + rank * FLAG_B + bid * 4, call);
    if (lane == 0) __hip_atomic_store(cnt, call, __ATOMIC_RELAXED, __HIP_MEMORY_SCOPE_AGENT);
  }
  const unsigned char* const xsrc = p.ws[srcr] + (size_t)(call & 1u) * SHARD_B;
  if (wid == 11) {
    if (!remote) return;
    const unsigned long long* fl = (const unsigned long long*)(myws + FLAG_OFF + srcr * FLAG_B) + lane * 2;
    unsigned* const tmo = (unsigned*)(myws + TMO_OFF);
    const unsigned long long t0 = __builtin_amdgcn_s_memrealtime();
    bool inv_ok = idx == INVIDX;
    for (unsigned it = 0;; ++it) {
      const unsigned long long f0 = __hip_atomic_load(fl, __ATOMIC_RELAXED, __HIP_MEMORY_SCOPE_SYSTEM);
      const unsigned long long f1 = __hip_atomic_load(fl + 1, __ATOMIC_RELAXED, __HIP_MEMORY_SCOPE_SYSTEM);
      const unsigned iv = inv_ok ? call : __hip_atomic_load(invw, __ATOMIC_RELAXED, __HIP_MEMORY_SCOPE_AGENT);
      const bool ok = (int)((unsigned)f0 - call) >= 0 && (int)((unsigned)(f0 >> 32) - call) >= 0 &&
                      (int)((unsigned)f1 - call) >= 0 && (int)((unsigned)(f1 >> 32) - call) >= 0;
      unsigned long long okm = __ballot(ok);
      inv_ok = (int)((unsigned)rfl((int)iv) - call) >= 0;
      if (!inv_ok && okm != 0ull && __builtin_amdgcn_s_memrealtime() - t0 > 500ull) {
        __builtin_amdgcn_fence(__ATOMIC_ACQUIRE, "");
        WAITV(0);
        inv_ok = true;
      }
      if ((it & 15u) == 15u &&
          (rfl((int)__hip_atomic_load(tmo, __ATOMIC_RELAXED, __HIP_MEMORY_SCOPE_AGENT)) == (int)call ||
           __builtin_amdgcn_s_memrealtime() - t0 > DEADLINE_TICKS)) {
        if (lane == 0) { __hip_atomic_store(tmo, call, __ATOMIC_RELAXED, __HIP_MEMORY_SCOPE_AGENT); lds_st(sync + 3, 1); }
        okm = ~0ull; inv_ok = true;
      }
      if (inv_ok) {
        if (lane == 0) { lds_st(sync + 12, (int)(unsigned)okm); lds_st(sync + 13, (int)(unsigned)(okm >> 32)); }
        if (okm == ~0ull) break;
      }
    }
    WAITV(0);
    return;
  }
  if (wid >= 8) {
    const int b = wid - 8;
    const int rstride = remote ? BK : K, sstride = remote ? STEP_B : BK;
    const unsigned char* xb = remote ? xsrc + (size_t)rb * BM * BK : px + (size_t)rb * BM * K;
    auto rx = __builtin_amdgcn_make_buffer_rsrc((void*)xb, 0, (int)SHARD_B - rb * BM * rstride, 0x00020000);
    auto rw = __builtin_amdgcn_make_buffer_rsrc((void*)(p.w + rank * NS + n0), 0, K * NT - (rank * NS + n0), 0x00020000);
    const int lr = lane >> 3, lc = lane & 7;
    const int sw0 = (lc ^ (lr >> 1)) << 4, sw1 = (lc ^ (4 + (lr >> 1))) << 4;
    const int xv0 = lr * rstride + sw0, xv1 = (lr + 8) * rstride + sw1 - 1024;
    const int wv0 = lr * NT + sw0, wv1 = (lr + 8) * NT + sw0 - 1024, wv2 = (lr + 16) * NT + sw1 - 2048, wv3 = (lr + 24) * NT + sw1 - 3072;
    const unsigned ldsb = ldsa + b * STAGE;
    const bool scout = remote && (tn & 1) == 0;
    for (int s = b; s < NSTEP; s += NSTAGE) {
      if (s >= NSTAGE) {
        for (unsigned it = 0; it < LDS_SPIN_CAP; ++it) {
          const int d = lds_ld(sync + 4 + (lane & 7));
          if (__all(d >= s - 2)) break;
          __builtin_amdgcn_s_sleep(1);
        }
      }
      const bool split = remote && s >= NSTEP - NSPLIT;
      const bool wlate = scout && s < NSTAGE;
      const int np = split ? 8 : 16;
#pragma unroll 1
      for (int job = wlate ? 1 : 0; job < (split ? 4 : 3); ++job) {
        if (job == 0 || (job == 2 && wlate)) {
#pragma unroll
          for (int qq = 0; qq < 4; ++qq) dma16x4(ldsb + XB + qq * 4096, wv0, wv1, wv2, wv3, rw, (s * BK + qq * 32) * NT);
        }
        if (job & 1) {
          const int turn = job == 1 ? s : s + NSPLIT;
          if (remote) {
            const int ub = 2 * s + rb;
            for (unsigned it = 0; it < LDS_SPIN_CAP; ++it) {
              const int v = lds_ld(sync + 12 + (lane & 3));
              const unsigned mw = (unsigned)__builtin_amdgcn_readlane(v, ub >> 5);
              if (((mw >> (ub & 31)) & 1u) && __builtin_amdgcn_readlane(v, 2) >= turn) break;
              __builtin_amdgcn_s_sleep(1);
            }
            if (SKEW > 0 && rb && s == 0) {
              const unsigned long long t1 = __builtin_amdgcn_s_memrealtime();
              while (__builtin_amdgcn_s_memrealtime() - t1 < SKEW) __builtin_amdgcn_s_sleep(1);
            }
          }
          const int pb = job == 1 ? 0 : 8;
#pragma unroll 2
          for (int pp = 0; pp < np; ++pp) {
            const int pr = pb + ((pp + tn) & (np - 1));
            dma16x2(ldsb + pr * 2048, xv0, xv1, rx, s * sstride + pr * 16 * rstride);
          }
          if (lane == 0) lds_st(sync + 14, turn + 1);
        }
        if (job >= 2) {
          WAITV(0);
          if (job == 2 ? (lane == 0 || (lane == 1 && !split)) : lane == 1) lds_st(sync + lane * 16 + b, s + 1);
        }
      }
    }
    return;
  }
  const int half = wid >> 2, wm = half * 2 + (wid & 1), wn = (wid >> 1) & 1;
  LAS int* const fullw = sync + half * 16;
  const int r = lane & 15, g = lane >> 4;
  const int xa0 = (wm * 64 + r) * 128 + ((g ^ ((r >> 1) & 7)) << 4);
  const int xa1 = xa0 ^ 64;
  const int fw = ((r >> 2) & 3) | ((g & 1) << 2);
  const int wa = XB + (16 * g + (r >> 1)) * 128 + (r & 1) * 8;
  int wan[4];
#pragma unroll
  for (int nt = 0; nt < 4; ++nt) wan[nt] = wa + (((wn * 4 + nt) ^ fw) << 4);
  v4f acc[4][4];
#pragma unroll
  for (int i = 0; i < 4; ++i)
#pragma unroll
    for (int j = 0; j < 4; ++j) acc[i][j] = (v4f){0.f, 0.f, 0.f, 0.f};
  float sc = p.sx[0] * p.sw[0];
  asm volatile("" : "+v"(sc));
  int buf = 0;
  int fullv = 0;
  for (int s = 0; s < NSTEP; ++s) {
    if (rfl(fullv) <= s) {
      for (unsigned it = 0; it < LDS_SPIN_CAP; ++it) {
        fullv = lds_ld(fullw + buf);
        if (rfl(fullv) > s) break;
        __builtin_amdgcn_s_sleep(1);
      }
    }
    asm volatile("" ::: "memory");
    const int sb = buf * STAGE;
    v8i xf[4], wf[4];
#pragma unroll
    for (int nt = 0; nt < 4; ++nt) {
      v2i q[4];
#pragma unroll
      for (int hs = 0; hs < 4; ++hs)
        q[hs] = __builtin_amdgcn_ds_read_tr8_b64_v2i32((LAS v2i*)(lds + sb + wan[nt] + (hs >> 1) * 8192 + (hs & 1) * 1024));
      wf[nt] = (v8i){q[0].x, q[0].y, q[1].x, q[1].y, q[2].x, q[2].y, q[3].x, q[3].y};
    }
#pragma unroll
    for (int mt = 0; mt < 4; ++mt) {
      v4i a = *(LAS v4i*)(lds + sb + xa0 + mt * 2048), c = *(LAS v4i*)(lds + sb + xa1 + mt * 2048);
      xf[mt] = (v8i){a.x, a.y, a.z, a.w, c.x, c.y, c.z, c.w};
    }
    buf = buf + 1; if (buf >= NSTAGE) buf = 0;
    __builtin_amdgcn_s_waitcnt(0xC07F);
    __builtin_amdgcn_sched_barrier(0);
    lds_st(sync + 4 + wid, s + 1);
    fullv = lds_ld(fullw + buf);
    __builtin_amdgcn_sched_barrier(0);
    __builtin_amdgcn_s_setprio(1);
#pragma unroll
    for (int mt = 0; mt < 4; ++mt)
#pragma unroll
      for (int nt = 0; nt < 4; ++nt)
        acc[mt][nt] = __builtin_amdgcn_mfma_scale_f32_16x16x128_f8f6f4(wf[nt], xf[mt], acc[mt][nt], 1, 0, 0, 0, 0, 0);
    __builtin_amdgcn_s_setprio(0);
    __builtin_amdgcn_sched_barrier(0);
  }
  for (unsigned it = 0; it < LDS_SPIN_CAP; ++it) {
    const int d = lds_ld(sync + 4 + half * 4 + (lane & 3));
    if (__all(d >= NSTEP - 1)) break;
    __builtin_amdgcn_s_sleep(1);
  }
  asm volatile("" ::: "memory");
  if (rfl(lds_ld(sync + 3)) != 0) sc = __builtin_nanf("");
  LAS unsigned char* const eb0 = lds + ((NSTEP - 3) % NSTAGE) * STAGE + wid * 4096;
  LAS unsigned char* const eb1 = lds + ((NSTEP - 2) % NSTAGE) * STAGE + wid * 4096;
  float* const o = p.out + (size_t)(m0 + wm * 64 + g) * NS + n0 + wn * 64 + r * 4;
  auto put = [&](int mt) {
    LAS unsigned char* eb = (mt & 1) ? eb1 : eb0;
#pragma unroll
    for (int nt = 0; nt < 4; ++nt)
      *(LAS v4f*)(eb + r * 256 + (((nt * 4 + g) ^ r) << 4)) =
          __builtin_elementwise_maximum(acc[mt][nt] * sc, (v4f){0.f, 0.f, 0.f, 0.f});
  };
  auto get = [&](int mt) {
    LAS unsigned char* eb = (mt & 1) ? eb1 : eb0;
    v4f v[4];
#pragma unroll
    for (int i = 0; i < 4; ++i) v[i] = *(LAS v4f*)(eb + (i * 4 + g) * 256 + ((r ^ (i * 4 + g)) << 4));
#pragma unroll
    for (int i = 0; i < 4; ++i) __builtin_nontemporal_store(v[i], (v4f*)(o + (size_t)(mt * 16 + i * 4) * NS));
  };
  put(0); put(1); get(0); put(2); get(1); put(3); get(2); get(3);
}
}

extern "C" void kernel_launch(int rank, int nranks, ncclComm_t comm,
                              hipStream_t stream,
                              void* const* d_in, const int* in_sizes, int n_in,
                              void* d_out, int out_size,
                              const int* in_dtypes, int out_dtype,
                              void* const* d_out_all,
                              void* const* d_ws_all, size_t ws_bytes) {
  (void)comm; (void)in_sizes; (void)n_in; (void)out_size; (void)in_dtypes; (void)out_dtype; (void)d_out_all;
  if (nranks != NR || (unsigned)rank >= (unsigned)NR || ws_bytes < WS_NEED) {
    fprintf(stderr, "unexpected configuration\n");
    return;
  }
  Params p;
  p.x = (const unsigned char*)d_in[0];
  p.w = (const unsigned char*)d_in[1];
  p.sx = (const float*)d_in[2];
  p.sw = (const float*)d_in[3];
  p.out = (float*)d_out;
  for (int i = 0; i < NR; ++i) p.ws[i] = (unsigned char*)d_ws_all[i];
  p.myws = p.ws[rank];
  p.rank = rank;
  p.pad = 0;
  HIP_CHECK(hipFuncSetAttribute((const void*)fused_ag_gemm, hipFuncAttributeMaxDynamicSharedMemorySize, LDS_BYTES));
  fused_ag_gemm<<<dim3(NWG), dim3(NTHREADS), LDS_BYTES, stream>>>(p);
  HIP_CHECK(hipGetLastError());
}
```
